```python
import numpy as np

kernel_name = "gpu_gemm_a2a_m2048_k2048_n2048_bf16_silu"


ABSMAX_RTOL = 0.008

COMPUTE_DTYPE = "bfloat16"
EPILOGUE = "silu"
IN_QUANT = None
OUT_QUANT = None
IN_DTYPES = {"x": "bfloat16", "w_mat": "bfloat16"}
WORLD_SIZE = 8

def _to_bf16(x):
    u = np.ascontiguousarray(x, dtype=np.float32).view(np.uint32)
    return ((u + 0x7FFF + ((u >> 16) & 1)) & 0xFFFF0000).view(np.float32)

def setup_inputs():

    x = _to_bf16(np.random.randn(2048, 2048).astype(np.float32))
    w_mat = _to_bf16((np.random.randn(2048, 2048) / np.sqrt(2048)).astype(np.float32))
    return {"x": x, "w_mat": w_mat}

def _epilogue(y):
    return (y / (1.0 + np.exp(-np.clip(y, -60.0, 60.0)))).astype(np.float32)

def reference(x, w_mat):
    y = (x @ w_mat).astype(np.float32)
    return _epilogue(y)
```

```cpp
#include <hip/hip_runtime.h>
#include <rccl/rccl.h>
#include <cstdio>
#include <cstdint>

#define HIP_CHECK(expr) do { hipError_t _e = (expr); if (_e != hipSuccess) { \
    fprintf(stderr, "HIP error %s at %s:%d\n", hipGetErrorString(_e), __FILE__, __LINE__); \
    return; } } while (0)

typedef unsigned u32x4 __attribute__((ext_vector_type(4)));
typedef short s16x4 __attribute__((ext_vector_type(4)));
typedef short s16x8 __attribute__((ext_vector_type(8)));
typedef __bf16 bf16x8 __attribute__((ext_vector_type(8)));
typedef __bf16 bf16x2 __attribute__((ext_vector_type(2)));
typedef float f32x2 __attribute__((ext_vector_type(2)));
typedef float f32x4 __attribute__((ext_vector_type(4)));
typedef float f32x16 __attribute__((ext_vector_type(16)));
#define LAS __attribute__((address_space(3)))
#define GAS __attribute__((address_space(1)))
#define CAS __attribute__((address_space(4)))

constexpr int NR = 8;
constexpr int MS = 256;
constexpr int KK = 2048;
constexpr int NN = 2048;
constexpr int NS = 256;
constexpr int STAGE_B = 12288;
constexpr int NSLOT = 3;
constexpr int WAVE_LDS = STAGE_B * NSLOT;
constexpr int LDS_BYTES = WAVE_LDS * 4;
constexpr int WS_FLAG = 0;
constexpr int WS_CNT = 4096;
constexpr int WS_TMO = 8192;
constexpr unsigned long long DEADLINE_TICKS = 50000000ull;

struct Args {
    const void* x;
    const void* w;
    char* out[NR];
    char* flag[NR];
    char* ws;
    int rank;
    int pad;
};

__device__ __forceinline__ void issue_stage(unsigned lds, unsigned vx0, unsigned vx1, unsigned vw,
                                            __amdgpu_buffer_rsrc_t rx, __amdgpu_buffer_rsrc_t rw,
                                            unsigned sx, unsigned sw) {
    unsigned t;
    asm volatile(
        "s_mov_b32 m0, %[lds]\n\t"
        "s_add_u32 %[t], %[sx], 31744\n\t"
        "buffer_load_dwordx4 %[vx0], %[rx], %[sx] offen lds\n\t"
        "buffer_load_dwordx4 %[vx1], %[rx], %[t] offen offset:1024 lds\n\t"
        "s_add_u32 %[t], %[t], 31744\n\t"
        "buffer_load_dwordx4 %[vx0], %[rx], %[t] offen offset:2048 lds\n\t"
        "s_add_u32 %[t], %[t], 31744\n\t"
        "buffer_load_dwordx4 %[vx1], %[rx], %[t] offen offset:3072 lds\n\t"
        "s_add_u32 m0, m0, 4096\n\t"
        "s_add_u32 %[t], %[sw], 31744\n\t"
        "buffer_load_dwordx4 %[vw], %[rw], %[sw] offen lds\n\t"
        "buffer_load_dwordx4 %[vw], %[rw], %[t] offen offset:1024 lds\n\t"
        "s_add_u32 %[t], %[t], 31744\n\t"
        "buffer_load_dwordx4 %[vw], %[rw], %[t] offen offset:2048 lds\n\t"
        "s_add_u32 %[t], %[t], 31744\n\t"
        "buffer_load_dwordx4 %[vw], %[rw], %[t] offen offset:3072 lds\n\t"
        "s_add_u32 m0, m0, 4096\n\t"
        "s_add_u32 %[t], %[t], 35840\n\t"
        "buffer_load_dwordx4 %[vw], %[rw], %[t] offen lds\n\t"
        "s_add_u32 %[t], %[t], 31744\n\t"
        "buffer_load_dwordx4 %[vw], %[rw], %[t] offen offset:1024 lds\n\t"
        "s_add_u32 %[t], %[t], 31744\n\t"
        "buffer_load_dwordx4 %[vw], %[rw], %[t] offen offset:2048 lds\n\t"
        "s_add_u32 %[t], %[t], 31744\n\t"
        "buffer_load_dwordx4 %[vw], %[rw], %[t] offen offset:3072 lds\n\t"
        : [t] "=&s"(t)
        : [lds] "s"(lds), [vx0] "v"(vx0), [vx1] "v"(vx1), [vw] "v"(vw), [rx] "s"(rx), [rw] "s"(rw),
          [sx] "s"(sx), [sw] "s"(sw)
        : "memory", "scc");
}

__device__ __forceinline__ float silu(float y) {
    float c = __builtin_amdgcn_fmed3f(y, -60.0f, 60.0f);
    float e = __builtin_amdgcn_exp2f(c * -1.44269504088896340736f);
    return y * __builtin_amdgcn_rcpf(1.0f + e);
}

__global__ __launch_bounds__(256, 1) void gemm_a2a(Args a) {
    extern __shared__ __attribute__((aligned(16))) char smem[];
    const int b = blockIdx.x;
    const int g = b & 7, idx = b >> 3;
    const int rt = idx & 7, ct = idx >> 3;
    const int p = (a.rank + g) & 7;
    const int tid = threadIdx.x;
    const int lane = tid & 63;
    const int wave = __builtin_amdgcn_readfirstlane(tid >> 6);

    const __amdgpu_buffer_rsrc_t rx = __builtin_amdgcn_make_buffer_rsrc((void*)a.x, (short)0, MS * KK * 2, 0x00020000);
    const __amdgpu_buffer_rsrc_t rw = __builtin_amdgcn_make_buffer_rsrc((void*)a.w, (short)0, KK * NN * 2, 0x00020000);

    const unsigned ldsbase = (unsigned)(uintptr_t)(LAS char*)smem + wave * WAVE_LDS;
    const unsigned lrow = (lane >> 3) * 4096;
    const unsigned vx0 = lrow + (((lane & 7) ^ (lane >> 4)) << 4);
    const unsigned vx1 = lrow + (((lane & 7) ^ (4 + (lane >> 4))) << 4);
    const unsigned vw = lrow + (((lane & 7) ^ (((lane >> 4) & 1) << 2)) << 4);
    unsigned sx = rt * 32 * 4096 + wave * 1024;
    unsigned sw = wave * 512 * 4096 + p * 512 + ct * 128;

    issue_stage(ldsbase, vx0, vx1, vw, rx, rw, sx, sw);
    issue_stage(ldsbase + STAGE_B, vx0, vx1, vw, rx, rw, sx + 128, sw + 64 * 4096);
    issue_stage(ldsbase + 2 * STAGE_B, vx0, vx1, vw, rx, rw, sx + 256, sw + 128 * 4096);

    const unsigned call = *(const CAS unsigned*)(uintptr_t)(a.ws + WS_CNT + b * 4) + 1;

    const int m = lane & 31, h = lane >> 5;
    const unsigned xa0 = ldsbase + m * 128 + ((h ^ ((m >> 1) & 7)) << 4);
    const int q = (lane >> 2) & 3, pp = lane & 3, blk = (lane >> 4) & 1;
    const unsigned wa0 = ldsbase + 4096 + (8 * h + q) * 128 + ((blk ^ (2 * (q >> 1))) << 5) + 8 * pp;

    f32x16 acc0 = {}, acc1 = {};
#pragma unroll
    for (int s = 0; s < 8; ++s) {
        if (s <= 5) asm volatile("s_waitcnt vmcnt(24)" ::: "memory");
        else if (s == 6) asm volatile("s_waitcnt vmcnt(12)" ::: "memory");
        else asm volatile("s_waitcnt vmcnt(0)" ::: "memory");
        const unsigned so = (s % NSLOT) * STAGE_B;
        bf16x8 xf[4], wf0[4], wf1[4];
#pragma unroll
        for (int j = 0; j < 4; ++j) {
            xf[j] = *(const LAS bf16x8*)(uintptr_t)((xa0 ^ (j * 32)) + so);
            s16x4 l0 = __builtin_amdgcn_ds_read_tr16_b64_v4i16((LAS s16x4*)(uintptr_t)(wa0 + so + j * 2048));
            s16x4 u0 = __builtin_amdgcn_ds_read_tr16_b64_v4i16((LAS s16x4*)(uintptr_t)(wa0 + so + j * 2048 + 512));
            s16x4 l1 = __builtin_amdgcn_ds_read_tr16_b64_v4i16((LAS s16x4*)(uintptr_t)((wa0 ^ 64) + so + j * 2048));
            s16x4 u1 = __builtin_amdgcn_ds_read_tr16_b64_v4i16((LAS s16x4*)(uintptr_t)((wa0 ^ 64) + so + j * 2048 + 512));
            wf0[j] = __builtin_bit_cast(bf16x8, __builtin_shufflevector(l0, u0, 0, 1, 2, 3, 4, 5, 6, 7));
            wf1[j] = __builtin_bit_cast(bf16x8, __builtin_shufflevector(l1, u1, 0, 1, 2, 3, 4, 5, 6, 7));
        }
        if (s + NSLOT < 8) {
            asm volatile("s_waitcnt lgkmcnt(0)" ::: "memory");
            issue_stage(ldsbase + so, vx0, vx1, vw, rx, rw, sx + (s + NSLOT) * 128, sw + (s + NSLOT) * 64 * 4096);
        }
#pragma unroll
        for (int j = 0; j < 4; ++j) {
            acc0 = __builtin_amdgcn_mfma_f32_32x32x16_bf16(wf0[j], xf[j], acc0, 0, 0, 0);
            acc1 = __builtin_amdgcn_mfma_f32_32x32x16_bf16(wf1[j], xf[j], acc1, 0, 0, 0);
        }
    }

    {
        const unsigned rb = ldsbase + m * 256;
#pragma unroll
        for (int r = 0; r < 4; ++r) {
            f32x4 v0 = {acc0[4 * r], acc0[4 * r + 1], acc0[4 * r + 2], acc0[4 * r + 3]};
            f32x4 v1 = {acc1[4 * r], acc1[4 * r + 1], acc1[4 * r + 2], acc1[4 * r + 3]};
            *(LAS f32x4*)(uintptr_t)(rb + (((2 * r + h) ^ (m & 7)) << 4)) = v0;
            *(LAS f32x4*)(uintptr_t)(rb + (((8 + 2 * r + h) ^ (m & 7)) << 4)) = v1;
        }
    }
    __syncthreads();
    {
        const int om = tid >> 3, oj = tid & 7;
        const unsigned base = (unsigned)(uintptr_t)(LAS char*)smem + om * 256;
        const unsigned o0 = base + (((2 * oj) ^ (om & 7)) << 4), o1 = base + (((2 * oj + 1) ^ (om & 7)) << 4);
        f32x4 lo = *(const LAS f32x4*)(uintptr_t)(o0), hi = *(const LAS f32x4*)(uintptr_t)(o1);
#pragma unroll
        for (int w2 = 1; w2 < 4; ++w2) {
            lo += *(const LAS f32x4*)(uintptr_t)(o0 + w2 * WAVE_LDS);
            hi += *(const LAS f32x4*)(uintptr_t)(o1 + w2 * WAVE_LDS);
        }
        u32x4 o;
        o[0] = __builtin_bit_cast(unsigned, __builtin_convertvector((f32x2){silu(lo[0]), silu(lo[1])}, bf16x2));
        o[1] = __builtin_bit_cast(unsigned, __builtin_convertvector((f32x2){silu(lo[2]), silu(lo[3])}, bf16x2));
        o[2] = __builtin_bit_cast(unsigned, __builtin_convertvector((f32x2){silu(hi[0]), silu(hi[1])}, bf16x2));
        o[3] = __builtin_bit_cast(unsigned, __builtin_convertvector((f32x2){silu(hi[2]), silu(hi[3])}, bf16x2));
        char* dst = a.out[g] + (rt * 32 + om) * (NS * 2) + ct * 128 + oj * 16;
        asm volatile("global_store_dwordx4 %0, %1, off sc0 sc1\n\ts_nop 1" :: "v"(dst), "v"(o) : "memory");
    }
    asm volatile("s_waitcnt vmcnt(0)" ::: "memory");
    __syncthreads();
    if (tid == 0) {
        if (g != 0)
            __hip_atomic_store((unsigned*)(a.flag[g] + idx * 4), call, __ATOMIC_RELAXED, __HIP_MEMORY_SCOPE_SYSTEM);
        *(unsigned*)(a.ws + WS_CNT + b * 4) = call;
    }
    if (b == 0 && wave == 0) {
        const char* fp = a.ws + WS_FLAG + lane * 16;
        const bool mine = (lane >> 3) == a.rank;
        const unsigned long long t0 = __builtin_amdgcn_s_memrealtime();
        for (;;) {
            u32x4 f;
            asm volatile("global_load_dwordx4 %0, %1, off sc0 sc1\n\ts_waitcnt vmcnt(0)" : "=&v"(f) : "v"(fp) : "memory");
            const bool ok = mine || (f[0] >= call && f[1] >= call && f[2] >= call && f[3] >= call);
            if (__all(ok)) break;
            if (__builtin_amdgcn_s_memrealtime() - t0 > DEADLINE_TICKS) {
                if (lane == 0) {
                    __hip_atomic_store((unsigned*)(a.ws + WS_TMO), call, __ATOMIC_RELAXED, __HIP_MEMORY_SCOPE_AGENT);
                    *(unsigned short*)a.out[0] = 0x7fc0;
                }
                break;
            }
            __builtin_amdgcn_s_sleep(1);
        }
    }
}

extern "C" void kernel_launch(int rank, int nranks, ncclComm_t comm,
                              hipStream_t stream,
                              void* const* d_in, const int* in_sizes, int n_in,
                              void* d_out, int out_size,
                              const int* in_dtypes, int out_dtype,
                              void* const* d_out_all,
                              void* const* d_ws_all, size_t ws_bytes) {
    if (nranks != NR || (unsigned)rank >= (unsigned)NR || n_in < 2 || in_sizes[0] != MS * KK ||
        in_sizes[1] != KK * NN || out_size != NN * NS || ws_bytes < 16384) {
        fprintf(stderr, "unexpected problem shape\n");
        return;
    }
    static bool init = false;
    if (!init) {
        HIP_CHECK(hipFuncSetAttribute((const void*)gemm_a2a, hipFuncAttributeMaxDynamicSharedMemorySize, LDS_BYTES));
        init = true;
    }
    Args a;
    a.x = d_in[0];
    a.w = d_in[1];
    for (int g = 0; g < NR; ++g) {
        const int p = (rank + g) & 7;
        a.out[g] = (char*)d_out_all[p] + (size_t)rank * MS * NS * 2;
        a.flag[g] = (char*)d_ws_all[p] + WS_FLAG + rank * 128;
    }
    a.ws = (char*)d_ws_all[rank];
    a.rank = rank;
    a.pad = 0;
    hipLaunchKernelGGL(gemm_a2a, dim3(256), dim3(256), LDS_BYTES, stream, a);
    HIP_CHECK(hipGetLastError());
}
```

```cpp
#include <hip/hip_runtime.h>
#include <rccl/rccl.h>
#include <cstdio>
#include <cstdint>
#include <cstddef>

#define HIP_CHECK(expr) do { hipError_t _e = (expr); if (_e != hipSuccess) { \
    fprintf(stderr, "HIP error %s at %s:%d\n", hipGetErrorString(_e), __FILE__, __LINE__); \
    return; } } while (0)

constexpr int NR = 8;
constexpr int MS = 256;
constexpr int KK = 2048;
constexpr int NN = 2048;
constexpr int NS = 256;
constexpr int LDS_BYTES = 163840;
constexpr int WS_FLAG = 16384;
constexpr size_t WS_USED = 65536 + 256 * 128;
constexpr int LOCAL_GROUP = 5;

struct Args {
    unsigned rx[4];
    unsigned rw[4];
    char* ws;
    int waiter;
    int cx;
    int slow;
    int pad[3];
    struct { char* out; char* flag; } dst[NR];
};
static_assert(offsetof(Args, ws) == 0x20 && offsetof(Args, slow) == 0x30 && offsetof(Args, dst) == 0x40, "layout");

__global__ __launch_bounds__(256, 1) void gemm_a2a(Args a) {
    __shared__ __attribute__((aligned(16))) char smem[LDS_BYTES];
    asm volatile(
        "s_load_dwordx16 s[4:19], %[ka], 0x0\n\t"
        "s_lshr_b32 s28, %[bid], 3\n\t"
        "v_readfirstlane_b32 s27, %[tid]\n\t"
        "s_getpc_b64 s[40:41]\n\t"
        "v_lshlrev_b32 v30, 6, %[tid]\n\t"
        "s_cmp_lg_u32 s28, 1\n\t"
        "s_cbranch_scc1 1f\n\t"
        "v_cmp_gt_u32 vcc, 9f-1f, v30\n\t"
        "s_and_b64 exec, exec, vcc\n\t"
        "s_cbranch_execz 1f\n\t"
        "global_load_dword v31, v30, s[40:41]\n\t"
        "1:\n\t"
        "s_mov_b64 exec, -1\n\t"
        "v_bfe_u32 v19, %[tid], 3, 3\n\t"
        "v_bfe_u32 v1, %[tid], 4, 2\n\t"
        "v_lshlrev_b32 v19, 12, v19\n\t"
        "v_bitop3_b32 v1, v1, %[tid], 7 bitop3:0x28\n\t"
        "v_lshrrev_b32 v2, 3, %[tid]\n\t"
        "v_and_b32 v2, 2, v2\n\t"
        "v_bitop3_b32 v2, v2, %[tid], 7 bitop3:0x28\n\t"
        "v_lshl_or_b32 v1, v1, 4, v19\n\t"
        "v_lshl_or_b32 v2, v2, 4, v19\n\t"
        "s_lshr_b32 s27, s27, 6\n\t"
        "s_and_b32 s33, s28, 7\n\t"
        "s_lshl_b32 s34, s33, 17\n\t"
        "s_lshl_b32 s29, s27, 7\n\t"
        "s_add_u32 s34, s34, s29\n\t"
        "v_add_u32 v1, s34, v1\n\t"
        "s_lshl_b32 s38, s27, 14\n\t"
        "s_add_u32 s38, s38, 98304\n\t"
        "s_movk_i32 s20, 31744\n\t"
        "s_and_b32 s35, %[bid], 7\n\t"
        "s_lshl_b32 s34, s35, 9\n\t"
        "s_lshr_b32 s36, s28, 3\n\t"
        "s_lshl_b32 s36, s36, 7\n\t"
        "s_add_u32 s34, s34, s36\n\t"
        "s_lshl_b32 s29, s27, 18\n\t"
        "s_add_u32 s34, s34, s29\n\t"
        "v_add_u32 v2, s34, v2\n\t"
        "s_mov_b32 s21, 63488\n\t"
        "s_mov_b32 s22, 95232\n\t"
        "s_mul_i32 s37, s27, 20480\n\t"
        "s_add_u32 s37, s37, 16384\n\t"
        "s_lshl_b32 s23, %[bid], 7\n\t"
        "s_add_u32 s23, s23, 65536\n\t"
        "s_lshl_b32 s24, s35, 4\n\t"
        "s_lshr_b32 s58, s28, 3\n\t"
        "s_add_u32 s59, s33, s58\n\t"
        "s_bfe_i32 s56, s59, 0x10001\n\t"
        "s_bfe_i32 s57, s33, 0x10002\n\t"
        "s_and_b32 s57, s57, 0x01010101\n\t"
        "s_and_b32 s29, %[bid], 7\n\t"
        "s_mul_i32 s39, s29, 0x11111111\n\t"
        "s_mul_i32 s43, s29, 0x8080808\n\t"
        "s_or_b32 s43, s43, s57\n\t"
        "s_and_b32 s29, %[bid], 1\n\t"
        "s_lshl2_add_u32 s58, s29, s58\n\t"
        "s_bitcmp1_b32 s59, 0\n\t"
        "s_cbranch_scc1 4f\n\t"
        "s_and_b32 s30, s56, 0x3736111\n\t"
        "s_xor_b32 s30, s30, 0x75316420\n\t"
        "s_xor_b32 s30, s30, s39\n\t"
        "s_and_b32 s44, s56, 0x39080808\n\t"
        "s_xor_b32 s44, s44, 0x30201000\n\t"
        "s_xor_b32 s44, s44, s43\n\t"
        "s_and_b32 s45, s56, 0x30213818\n\t"
        "s_xor_b32 s45, s45, 0x31211101\n\t"
        "s_xor_b32 s45, s45, s43\n\t"
        "s_and_b32 s46, s56, 0x38381818\n\t"
        "s_xor_b32 s46, s46, 0x19180908\n\t"
        "s_xor_b32 s46, s46, s43\n\t"
        "s_and_b32 s47, s56, 0x1818\n\t"
        "s_xor_b32 s47, s47, 0x39382928\n\t"
        "s_xor_b32 s47, s47, s43\n\t"
        "s_and_b32 s48, s56, 0x61000000\n\t"
        "s_xor_b32 s48, s48, 0x60402000\n\t"
        "s_xor_b32 s48, s48, s57\n\t"
        "s_and_b32 s49, s56, 0x216020\n\t"
        "s_xor_b32 s49, s49, 0x61412101\n\t"
        "s_xor_b32 s49, s49, s57\n\t"
        "s_xor_b32 s50, s57, 0xa1a08180\n\t"
        "s_xor_b32 s51, s57, 0xe1e0c1c0\n\t"
        "s_mov_b32 s58, -1\n\t"
        "s_branch 5f\n\t"
        "4:\n\t"
        "s_and_b32 s30, s56, 0x13200000\n\t"
        "s_xor_b32 s30, s30, 0x76543210\n\t"
        "s_xor_b32 s30, s30, s39\n\t"
        "s_xor_b32 s44, s43, 0x9080100\n\t"
        "s_xor_b32 s45, s43, 0x19181110\n\t"
        "s_and_b32 s46, s56, 0x10100000\n\t"
        "s_xor_b32 s46, s46, 0x29282120\n\t"
        "s_xor_b32 s46, s46, s43\n\t"
        "s_and_b32 s47, s56, 0x8081818\n\t"
        "s_xor_b32 s47, s47, 0x39383130\n\t"
        "s_xor_b32 s47, s47, s43\n\t"
        "s_xor_b32 s48, s57, 0x21200100\n\t"
        "s_xor_b32 s49, s57, 0x61604140\n\t"
        "s_xor_b32 s50, s57, 0xa1a08180\n\t"
        "s_xor_b32 s51, s57, 0xe1e0c1c0\n\t"
        "5:\n\t"
        "s_bfe_u32 s29, s30, 0x30000\n\t"
        "v_lshl_add_u32 v3, s29, 9, v1\n\t"
        "s_mov_b32 m0, s38\n\t"
        "v_xor_b32 v4, 64, v3\n\t"
        "s_waitcnt lgkmcnt(0)\n\t"
        "buffer_load_dwordx4 v3, s[4:7], 0 offen lds\n\t"
        "buffer_load_dwordx4 v4, s[4:7], s20 offen offset:1024 lds\n\t"
        "s_load_dword s31, s[12:13], s23\n\t"
        "s_load_dwordx4 s[52:55], %[ka], s24 offset:0x40\n\t"
        "v_xor_b32 v2, s15, v2\n\t"
        "s_bfe_u32 s29, s44, 0x80000\n\t"
        "v_lshl_add_u32 v5, s29, 17, v2\n\t"
        "s_mov_b32 m0, s37\n\t"
        "v_xor_b32 v24, 64, v5\n\t"
        "buffer_load_dwordx4 v5, s[8:11], 0 offen offset:0 lds\n\t"
        "buffer_load_dwordx4 v24, s[8:11], s20 offen offset:1024 lds\n\t"
        "buffer_load_dwordx4 v5, s[8:11], s21 offen offset:2048 lds\n\t"
        "buffer_load_dwordx4 v24, s[8:11], s22 offen offset:3072 lds\n\t"
        "s_xor_b32 s29, s28, 1\n\t"
        "s_or_b32 s29, s29, s27\n\t"
        "s_cbranch_scc1 1f\n\t"
        "s_mov_b32 s29, 0x0\n\t"
        "s_load_dword s26, s[8:9], s29\n\t"
        "s_mov_b32 s29, 0x100000\n\t"
        "s_load_dword s26, s[8:9], s29\n\t"
        "s_mov_b32 s29, 0x200000\n\t"
        "s_load_dword s26, s[8:9], s29\n\t"
        "s_mov_b32 s29, 0x300000\n\t"
        "s_load_dword s26, s[8:9], s29\n\t"
        "s_mov_b32 s29, 0x400000\n\t"
        "s_load_dword s26, s[8:9], s29\n\t"
        "s_mov_b32 s29, 0x500000\n\t"
        "s_load_dword s26, s[8:9], s29\n\t"
        "s_mov_b32 s29, 0x600000\n\t"
        "s_load_dword s26, s[8:9], s29\n\t"
        "s_mov_b32 s29, 0x700000\n\t"
        "s_load_dword s26, s[8:9], s29\n\t"
        "1:\n\t"
        "s_cmp_lg_u32 s58, s16\n\t"
        "s_cbranch_scc1 3f\n\t"
        "s_and_b32 s30, s56, 0x11000110\n\t"
        "s_xor_b32 s30, s30, 0x75321640\n\t"
        "s_xor_b32 s30, s30, s39\n\t"
        "s_and_b32 s44, s56, 0x80800\n\t"
        "s_xor_b32 s44, s44, 0x1302000\n\t"
        "s_xor_b32 s44, s44, s43\n\t"
        "s_and_b32 s45, s56, 0x808\n\t"
        "s_xor_b32 s45, s45, 0x9083121\n\t"
        "s_xor_b32 s45, s45, s43\n\t"
        "s_xor_b32 s46, s43, 0x19181110\n\t"
        "s_and_b32 s47, s56, 0x8080808\n\t"
        "s_xor_b32 s47, s47, 0x39382928\n\t"
        "s_xor_b32 s47, s47, s43\n\t"
        "s_xor_b32 s48, s57, 0x1402000\n\t"
        "s_xor_b32 s49, s57, 0x61604121\n\t"
        "3:\n\t"
        "s_bfe_u32 s29, s30, 0x30004\n\t"
        "v_lshl_add_u32 v3, s29, 9, v1\n\t"
        "s_add_u32 m0, s38, 2048\n\t"
        "v_xor_b32 v4, 64, v3\n\t"
        "buffer_load_dwordx4 v3, s[4:7], 0 offen lds\n\t"
        "buffer_load_dwordx4 v4, s[4:7], s20 offen offset:1024 lds\n\t"
        "s_bfe_u32 s29, s44, 0x80008\n\t"
        "v_lshl_add_u32 v5, s29, 17, v2\n\t"
        "s_add_u32 m0, s37, 4096\n\t"
        "v_xor_b32 v24, 64, v5\n\t"
        "buffer_load_dwordx4 v5, s[8:11], 0 offen offset:0 lds\n\t"
        "buffer_load_dwordx4 v24, s[8:11], s20 offen offset:1024 lds\n\t"
        "buffer_load_dwordx4 v5, s[8:11], s21 offen offset:2048 lds\n\t"
        "buffer_load_dwordx4 v24, s[8:11], s22 offen offset:3072 lds\n\t"
        "v_mov_b32 v22, s23\n\t"
        "v_add_u32 v25, 0x10000, v1\n\t"
        "v_and_b32 v19, 15, %[tid]\n\t"
        "v_bfe_u32 v20, %[tid], 4, 2\n\t"
        "v_lshrrev_b32 v21, 1, v19\n\t"
        "v_xor_b32 v21, v21, v20\n\t"
        "v_lshlrev_b32 v6, 7, v19\n\t"
        "v_lshl_or_b32 v6, v21, 4, v6\n\t"
        "v_add_u32 v6, s38, v6\n\t"
        "s_bfe_u32 s29, s48, 0x80000\n\t"
        "s_lshl_b32 s29, s29, 6\n\t"
        "v_xor_b32 v128, s29, v6\n\t"
        "s_bfe_u32 s29, s48, 0x80008\n\t"
        "s_lshl_b32 s29, s29, 6\n\t"
        "v_xor_b32 v129, s29, v6\n\t"
        "s_bfe_u32 s29, s48, 0x80010\n\t"
        "s_lshl_b32 s29, s29, 6\n\t"
        "v_xor_b32 v130, s29, v6\n\t"
        "s_bfe_u32 s29, s48, 0x80018\n\t"
        "s_lshl_b32 s29, s29, 6\n\t"
        "v_xor_b32 v131, s29, v6\n\t"
        "s_bfe_u32 s29, s49, 0x80000\n\t"
        "s_lshl_b32 s29, s29, 6\n\t"
        "v_xor_b32 v132, s29, v6\n\t"
        "s_bfe_u32 s29, s49, 0x80008\n\t"
        "s_lshl_b32 s29, s29, 6\n\t"
        "v_xor_b32 v133, s29, v6\n\t"
        "s_bfe_u32 s29, s49, 0x80010\n\t"
        "s_lshl_b32 s29, s29, 6\n\t"
        "v_xor_b32 v134, s29, v6\n\t"
        "s_bfe_u32 s29, s49, 0x80018\n\t"
        "s_lshl_b32 s29, s29, 6\n\t"
        "v_xor_b32 v135, s29, v6\n\t"
        "s_bfe_u32 s29, s50, 0x80000\n\t"
        "s_lshl_b32 s29, s29, 6\n\t"
        "v_xor_b32 v136, s29, v6\n\t"
        "s_bfe_u32 s29, s50, 0x80008\n\t"
        "s_lshl_b32 s29, s29, 6\n\t"
        "v_xor_b32 v137, s29, v6\n\t"
        "s_bfe_u32 s29, s50, 0x80010\n\t"
        "s_lshl_b32 s29, s29, 6\n\t"
        "v_xor_b32 v138, s29, v6\n\t"
        "s_bfe_u32 s29, s50, 0x80018\n\t"
        "s_lshl_b32 s29, s29, 6\n\t"
        "v_xor_b32 v139, s29, v6\n\t"
        "s_bfe_u32 s29, s51, 0x80000\n\t"
        "s_lshl_b32 s29, s29, 6\n\t"
        "v_xor_b32 v140, s29, v6\n\t"
        "s_bfe_u32 s29, s51, 0x80008\n\t"
        "s_lshl_b32 s29, s29, 6\n\t"
        "v_xor_b32 v141, s29, v6\n\t"
        "s_bfe_u32 s29, s51, 0x80010\n\t"
        "s_lshl_b32 s29, s29, 6\n\t"
        "v_xor_b32 v142, s29, v6\n\t"
        "s_bfe_u32 s29, s51, 0x80018\n\t"
        "s_lshl_b32 s29, s29, 6\n\t"
        "v_xor_b32 v143, s29, v6\n\t"
        "v_bfe_u32 v21, %[tid], 2, 2\n\t"
        "v_lshl_or_b32 v8, v20, 3, v21\n\t"
        "v_lshlrev_b32 v8, 7, v8\n\t"
        "v_bfe_u32 v23, %[tid], 3, 2\n\t"
        "v_lshl_or_b32 v8, v23, 5, v8\n\t"
        "v_and_b32 v23, 3, %[tid]\n\t"
        "v_lshl_or_b32 v8, v23, 3, v8\n\t"
        "v_add_u32 v8, s37, v8\n\t"
        "v_xor_b32 v9, 32, v8\n\t"
        "v_xor_b32 v10, 64, v8\n\t"
        "v_xor_b32 v11, 0x60, v8\n\t"
        "s_lshl_b32 s29, s27, 12\n\t"
        "v_bitop3_b32 v21, v20, %[tid], 7 bitop3:0x28\n\t"
        "v_lshlrev_b32 v12, 8, v19\n\t"
        "v_lshl_or_b32 v12, v21, 4, v12\n\t"
        "v_add_u32 v12, s29, v12\n\t"
        "v_xor_b32 v13, 64, v12\n\t"
        "s_lshl_b32 s29, s27, 2\n\t"
        "v_bfe_u32 v21, %[tid], 3, 2\n\t"
        "v_or_b32 v21, s29, v21\n\t"
        "v_and_b32 v17, 7, %[tid]\n\t"
        "v_bfe_u32 v23, %[tid], 5, 1\n\t"
        "v_lshl_or_b32 v23, v17, 1, v23\n\t"
        "v_bitop3_b32 v23, v23, v21, 7 bitop3:0x78\n\t"
        "v_lshlrev_b32 v16, 8, v21\n\t"
        "v_lshl_or_b32 v16, v23, 4, v16\n\t"
        "s_lshl_b32 s29, s33, 14\n\t"
        "s_add_u32 s29, s29, s36\n\t"
        "v_lshlrev_b32 v18, 9, v21\n\t"
        "v_lshl_or_b32 v18, v17, 4, v18\n\t"
        "v_add_u32 v18, s29, v18\n\t"
        "v_add_u32 v19, 0x2000, v18\n\t"
        "s_lshl_b32 s29, s28, 10\n\t"
        "v_mov_b32 v21, s29\n\t"
        "s_mov_b32 s40, 0xbfb8aa3b\n\t"
        "s_mov_b32 s41, 0xbfb8aa3b\n\t"
        "s_mov_b32 s42, 0x42ad1f97\n\t"
        "s_movk_i32 s25, 8192\n\t"
        "s_bfe_u32 s29, s30, 0x30008\n\t"
        "v_lshl_add_u32 v3, s29, 9, v1\n\t"
        "s_add_u32 m0, s38, 4096\n\t"
        "v_xor_b32 v4, 64, v3\n\t"
        "buffer_load_dwordx4 v3, s[4:7], 0 offen lds\n\t"
        "buffer_load_dwordx4 v4, s[4:7], s20 offen offset:1024 lds\n\t"
        "s_bfe_u32 s29, s44, 0x80010\n\t"
        "v_lshl_add_u32 v5, s29, 17, v2\n\t"
        "s_add_u32 m0, s37, 8192\n\t"
        "v_xor_b32 v24, 64, v5\n\t"
        "buffer_load_dwordx4 v5, s[8:11], 0 offen offset:0 lds\n\t"
        "buffer_load_dwordx4 v24, s[8:11], s20 offen offset:1024 lds\n\t"
        "buffer_load_dwordx4 v5, s[8:11], s21 offen offset:2048 lds\n\t"
        "buffer_load_dwordx4 v24, s[8:11], s22 offen offset:3072 lds\n\t"
        "s_waitcnt vmcnt(12)\n\t"
        "ds_read_b128 v[32:35], v128\n\t"
        "ds_read_b64_tr_b16 a[0:1], v8 offset:0\n\t"
        "ds_read_b64_tr_b16 a[2:3], v8 offset:512\n\t"
        "ds_read_b64_tr_b16 a[4:5], v9 offset:0\n\t"
        "ds_read_b64_tr_b16 a[6:7], v9 offset:512\n\t"
        "ds_read_b64_tr_b16 a[8:9], v10 offset:0\n\t"
        "ds_read_b64_tr_b16 a[10:11], v10 offset:512\n\t"
        "ds_read_b64_tr_b16 a[12:13], v11 offset:0\n\t"
        "ds_read_b64_tr_b16 a[14:15], v11 offset:512\n\t"
        "s_bfe_u32 s29, s30, 0x3000c\n\t"
        "v_lshl_add_u32 v3, s29, 9, v1\n\t"
        "s_add_u32 m0, s38, 6144\n\t"
        "v_xor_b32 v4, 64, v3\n\t"
        "buffer_load_dwordx4 v3, s[4:7], 0 offen lds\n\t"
        "buffer_load_dwordx4 v4, s[4:7], s20 offen offset:1024 lds\n\t"
        "s_waitcnt lgkmcnt(0)\n\t"
        "v_mfma_f32_16x16x32_bf16 v[80:83], a[0:3], v[32:35], 0\n\t"
        "v_mfma_f32_16x16x32_bf16 v[84:87], a[4:7], v[32:35], 0\n\t"
        "v_mfma_f32_16x16x32_bf16 v[88:91], a[8:11], v[32:35], 0\n\t"
        "v_mfma_f32_16x16x32_bf16 v[92:95], a[12:15], v[32:35], 0\n\t"
        "s_add_u32 s31, s31, 1\n\t"
        "v_mov_b32 v20, s31\n\t"
        "s_bfe_u32 s29, s44, 0x80018\n\t"
        "v_lshl_add_u32 v5, s29, 17, v2\n\t"
        "s_add_u32 m0, s37, 12288\n\t"
        "v_xor_b32 v24, 64, v5\n\t"
        "buffer_load_dwordx4 v5, s[8:11], 0 offen offset:0 lds\n\t"
        "buffer_load_dwordx4 v24, s[8:11], s20 offen offset:1024 lds\n\t"
        "buffer_load_dwordx4 v5, s[8:11], s21 offen offset:2048 lds\n\t"
        "buffer_load_dwordx4 v24, s[8:11], s22 offen offset:3072 lds\n\t"
        "s_waitcnt vmcnt(12)\n\t"
        "ds_read_b128 v[36:39], v129\n\t"
        "ds_read_b64_tr_b16 a[16:17], v8 offset:4096\n\t"
        "ds_read_b64_tr_b16 a[18:19], v8 offset:4608\n\t"
        "ds_read_b64_tr_b16 a[20:21], v9 offset:4096\n\t"
        "ds_read_b64_tr_b16 a[22:23], v9 offset:4608\n\t"
        "ds_read_b64_tr_b16 a[24:25], v10 offset:4096\n\t"
        "ds_read_b64_tr_b16 a[26:27], v10 offset:4608\n\t"
        "ds_read_b64_tr_b16 a[28:29], v11 offset:4096\n\t"
        "ds_read_b64_tr_b16 a[30:31], v11 offset:4608\n\t"
        "s_barrier\n\t"
        "s_bfe_u32 s29, s45, 0x80000\n\t"
        "v_lshl_add_u32 v5, s29, 17, v2\n\t"
        "s_add_u32 m0, s37, 16384\n\t"
        "v_xor_b32 v24, 64, v5\n\t"
        "buffer_load_dwordx4 v5, s[8:11], 0 offen offset:0 lds\n\t"
        "buffer_load_dwordx4 v24, s[8:11], s20 offen offset:1024 lds\n\t"
        "s_waitcnt lgkmcnt(0)\n\t"
        "v_mfma_f32_16x16x32_bf16 v[80:83], a[16:19], v[36:39], v[80:83]\n\t"
        "v_mfma_f32_16x16x32_bf16 v[84:87], a[20:23], v[36:39], v[84:87]\n\t"
        "v_mfma_f32_16x16x32_bf16 v[88:91], a[24:27], v[36:39], v[88:91]\n\t"
        "v_mfma_f32_16x16x32_bf16 v[92:95], a[28:31], v[36:39], v[92:95]\n\t"
        "buffer_load_dwordx4 v5, s[8:11], s21 offen offset:2048 lds\n\t"
        "buffer_load_dwordx4 v24, s[8:11], s22 offen offset:3072 lds\n\t"
        "s_bfe_u32 s29, s45, 0x80008\n\t"
        "v_lshl_add_u32 v5, s29, 17, v2\n\t"
        "s_mov_b32 m0, s37\n\t"
        "v_xor_b32 v24, 64, v5\n\t"
        "buffer_load_dwordx4 v5, s[8:11], 0 offen offset:0 lds\n\t"
        "buffer_load_dwordx4 v24, s[8:11], s20 offen offset:1024 lds\n\t"
        "s_waitcnt vmcnt(12)\n\t"
        "ds_read_b128 v[40:43], v130\n\t"
        "ds_read_b64_tr_b16 a[32:33], v8 offset:8192\n\t"
        "ds_read_b64_tr_b16 a[34:35], v8 offset:8704\n\t"
        "ds_read_b64_tr_b16 a[36:37], v9 offset:8192\n\t"
        "ds_read_b64_tr_b16 a[38:39], v9 offset:8704\n\t"
        "ds_read_b64_tr_b16 a[40:41], v10 offset:8192\n\t"
        "ds_read_b64_tr_b16 a[42:43], v10 offset:8704\n\t"
        "ds_read_b64_tr_b16 a[44:45], v11 offset:8192\n\t"
        "ds_read_b64_tr_b16 a[46:47], v11 offset:8704\n\t"
        "buffer_load_dwordx4 v5, s[8:11], s21 offen offset:2048 lds\n\t"
        "buffer_load_dwordx4 v24, s[8:11], s22 offen offset:3072 lds\n\t"
        "s_waitcnt lgkmcnt(0)\n\t"
        "v_mfma_f32_16x16x32_bf16 v[80:83], a[32:35], v[40:43], v[80:83]\n\t"
        "v_mfma_f32_16x16x32_bf16 v[84:87], a[36:39], v[40:43], v[84:87]\n\t"
        "v_mfma_f32_16x16x32_bf16 v[88:91], a[40:43], v[40:43], v[88:91]\n\t"
        "v_mfma_f32_16x16x32_bf16 v[92:95], a[44:47], v[40:43], v[92:95]\n\t"
        "s_barrier\n\t"
        "s_bfe_u32 s29, s45, 0x80010\n\t"
        "v_lshl_add_u32 v5, s29, 17, v2\n\t"
        "s_add_u32 m0, s37, 4096\n\t"
        "v_xor_b32 v24, 64, v5\n\t"
        "buffer_load_dwordx4 v5, s[8:11], 0 offen offset:0 lds\n\t"
        "buffer_load_dwordx4 v24, s[8:11], s20 offen offset:1024 lds\n\t"
        "buffer_load_dwordx4 v5, s[8:11], s21 offen offset:2048 lds\n\t"
        "buffer_load_dwordx4 v24, s[8:11], s22 offen offset:3072 lds\n\t"
        "s_waitcnt vmcnt(12)\n\t"
        "ds_read_b128 v[44:47], v131\n\t"
        "ds_read_b64_tr_b16 a[48:49], v8 offset:12288\n\t"
        "ds_read_b64_tr_b16 a[50:51], v8 offset:12800\n\t"
        "ds_read_b64_tr_b16 a[52:53], v9 offset:12288\n\t"
        "ds_read_b64_tr_b16 a[54:55], v9 offset:12800\n\t"
        "ds_read_b64_tr_b16 a[56:57], v10 offset:12288\n\t"
        "ds_read_b64_tr_b16 a[58:59], v10 offset:12800\n\t"
        "ds_read_b64_tr_b16 a[60:61], v11 offset:12288\n\t"
        "ds_read_b64_tr_b16 a[62:63], v11 offset:12800\n\t"
        "s_bfe_u32 s29, s45, 0x80018\n\t"
        "v_lshl_add_u32 v5, s29, 17, v2\n\t"
        "s_add_u32 m0, s37, 8192\n\t"
        "v_xor_b32 v24, 64, v5\n\t"
        "buffer_load_dwordx4 v5, s[8:11], 0 offen offset:0 lds\n\t"
        "buffer_load_dwordx4 v24, s[8:11], s20 offen offset:1024 lds\n\t"
        "s_waitcnt lgkmcnt(0)\n\t"
        "v_mfma_f32_16x16x32_bf16 v[80:83], a[48:51], v[44:47], v[80:83]\n\t"
        "v_mfma_f32_16x16x32_bf16 v[84:87], a[52:55], v[44:47], v[84:87]\n\t"
        "v_mfma_f32_16x16x32_bf16 v[88:91], a[56:59], v[44:47], v[88:91]\n\t"
        "v_mfma_f32_16x16x32_bf16 v[92:95], a[60:63], v[44:47], v[92:95]\n\t"
        "buffer_load_dwordx4 v5, s[8:11], s21 offen offset:2048 lds\n\t"
        "buffer_load_dwordx4 v24, s[8:11], s22 offen offset:3072 lds\n\t"
        "s_waitcnt vmcnt(12)\n\t"
        "ds_read_b128 v[48:51], v132\n\t"
        "ds_read_b64_tr_b16 a[64:65], v8 offset:16384\n\t"
        "ds_read_b64_tr_b16 a[66:67], v8 offset:16896\n\t"
        "ds_read_b64_tr_b16 a[68:69], v9 offset:16384\n\t"
        "ds_read_b64_tr_b16 a[70:71], v9 offset:16896\n\t"
        "ds_read_b64_tr_b16 a[72:73], v10 offset:16384\n\t"
        "ds_read_b64_tr_b16 a[74:75], v10 offset:16896\n\t"
        "ds_read_b64_tr_b16 a[76:77], v11 offset:16384\n\t"
        "ds_read_b64_tr_b16 a[78:79], v11 offset:16896\n\t"
        "s_barrier\n\t"
        "s_bfe_u32 s29, s30, 0x30010\n\t"
        "v_lshl_add_u32 v3, s29, 9, v1\n\t"
        "s_add_u32 m0, s38, 8192\n\t"
        "v_xor_b32 v4, 64, v3\n\t"
        "buffer_load_dwordx4 v3, s[4:7], 0 offen lds\n\t"
        "buffer_load_dwordx4 v4, s[4:7], s20 offen offset:1024 lds\n\t"
        "s_waitcnt lgkmcnt(0)\n\t"
        "v_mfma_f32_16x16x32_bf16 v[80:83], a[64:67], v[48:51], v[80:83]\n\t"
        "v_mfma_f32_16x16x32_bf16 v[84:87], a[68:71], v[48:51], v[84:87]\n\t"
        "v_mfma_f32_16x16x32_bf16 v[88:91], a[72:75], v[48:51], v[88:91]\n\t"
        "v_mfma_f32_16x16x32_bf16 v[92:95], a[76:79], v[48:51], v[92:95]\n\t"
        "s_bfe_u32 s29, s46, 0x80000\n\t"
        "v_lshl_add_u32 v5, s29, 17, v2\n\t"
        "s_add_u32 m0, s37, 12288\n\t"
        "v_xor_b32 v24, 64, v5\n\t"
        "buffer_load_dwordx4 v5, s[8:11], 0 offen offset:0 lds\n\t"
        "buffer_load_dwordx4 v24, s[8:11], s20 offen offset:1024 lds\n\t"
        "s_waitcnt vmcnt(12)\n\t"
        "ds_read_b128 v[52:55], v133\n\t"
        "ds_read_b64_tr_b16 a[80:81], v8 offset:0\n\t"
        "ds_read_b64_tr_b16 a[82:83], v8 offset:512\n\t"
        "ds_read_b64_tr_b16 a[84:85], v9 offset:0\n\t"
        "ds_read_b64_tr_b16 a[86:87], v9 offset:512\n\t"
        "ds_read_b64_tr_b16 a[88:89], v10 offset:0\n\t"
        "ds_read_b64_tr_b16 a[90:91], v10 offset:512\n\t"
        "ds_read_b64_tr_b16 a[92:93], v11 offset:0\n\t"
        "ds_read_b64_tr_b16 a[94:95], v11 offset:512\n\t"
        "buffer_load_dwordx4 v5, s[8:11], s21 offen offset:2048 lds\n\t"
        "buffer_load_dwordx4 v24, s[8:11], s22 offen offset:3072 lds\n\t"
        "s_waitcnt lgkmcnt(0)\n\t"
        "v_mfma_f32_16x16x32_bf16 v[80:83], a[80:83], v[52:55], v[80:83]\n\t"
        "v_mfma_f32_16x16x32_bf16 v[84:87], a[84:87], v[52:55], v[84:87]\n\t"
        "v_mfma_f32_16x16x32_bf16 v[88:91], a[88:91], v[52:55], v[88:91]\n\t"
        "v_mfma_f32_16x16x32_bf16 v[92:95], a[92:95], v[52:55], v[92:95]\n\t"
        "s_bfe_u32 s29, s46, 0x80008\n\t"
        "v_lshl_add_u32 v5, s29, 17, v2\n\t"
        "s_add_u32 m0, s37, 16384\n\t"
        "v_xor_b32 v24, 64, v5\n\t"
        "buffer_load_dwordx4 v5, s[8:11], 0 offen offset:0 lds\n\t"
        "buffer_load_dwordx4 v24, s[8:11], s20 offen offset:1024 lds\n\t"
        "s_waitcnt vmcnt(12)\n\t"
        "ds_read_b128 v[56:59], v134\n\t"
        "ds_read_b64_tr_b16 a[96:97], v8 offset:4096\n\t"
        "ds_read_b64_tr_b16 a[98:99], v8 offset:4608\n\t"
        "ds_read_b64_tr_b16 a[100:101], v9 offset:4096\n\t"
        "ds_read_b64_tr_b16 a[102:103], v9 offset:4608\n\t"
        "ds_read_b64_tr_b16 a[104:105], v10 offset:4096\n\t"
        "ds_read_b64_tr_b16 a[106:107], v10 offset:4608\n\t"
        "ds_read_b64_tr_b16 a[108:109], v11 offset:4096\n\t"
        "ds_read_b64_tr_b16 a[110:111], v11 offset:4608\n\t"
        "buffer_load_dwordx4 v5, s[8:11], s21 offen offset:2048 lds\n\t"
        "buffer_load_dwordx4 v24, s[8:11], s22 offen offset:3072 lds\n\t"
        "s_waitcnt lgkmcnt(0)\n\t"
        "v_mfma_f32_16x16x32_bf16 v[80:83], a[96:99], v[56:59], v[80:83]\n\t"
        "v_mfma_f32_16x16x32_bf16 v[84:87], a[100:103], v[56:59], v[84:87]\n\t"
        "v_mfma_f32_16x16x32_bf16 v[88:91], a[104:107], v[56:59], v[88:91]\n\t"
        "v_mfma_f32_16x16x32_bf16 v[92:95], a[108:111], v[56:59], v[92:95]\n\t"
        "s_barrier\n\t"
        "s_bfe_u32 s29, s30, 0x30014\n\t"
        "v_lshl_add_u32 v3, s29, 9, v1\n\t"
        "s_add_u32 m0, s38, 10240\n\t"
        "v_xor_b32 v4, 64, v3\n\t"
        "buffer_load_dwordx4 v3, s[4:7], 0 offen lds\n\t"
        "buffer_load_dwordx4 v4, s[4:7], s20 offen offset:1024 lds\n\t"
        "s_waitcnt vmcnt(12)\n\t"
        "ds_read_b128 v[60:63], v135\n\t"
        "ds_read_b64_tr_b16 a[112:113], v8 offset:8192\n\t"
        "ds_read_b64_tr_b16 a[114:115], v8 offset:8704\n\t"
        "ds_read_b64_tr_b16 a[116:117], v9 offset:8192\n\t"
        "ds_read_b64_tr_b16 a[118:119], v9 offset:8704\n\t"
        "ds_read_b64_tr_b16 a[120:121], v10 offset:8192\n\t"
        "ds_read_b64_tr_b16 a[122:123], v10 offset:8704\n\t"
        "ds_read_b64_tr_b16 a[124:125], v11 offset:8192\n\t"
        "ds_read_b64_tr_b16 a[126:127], v11 offset:8704\n\t"
        "s_bfe_u32 s29, s46, 0x80010\n\t"
        "v_lshl_add_u32 v5, s29, 17, v2\n\t"
        "s_mov_b32 m0, s37\n\t"
        "v_xor_b32 v24, 64, v5\n\t"
        "buffer_load_dwordx4 v5, s[8:11], 0 offen offset:0 lds\n\t"
        "buffer_load_dwordx4 v24, s[8:11], s20 offen offset:1024 lds\n\t"
        "s_waitcnt lgkmcnt(0)\n\t"
        "v_mfma_f32_16x16x32_bf16 v[80:83], a[112:115], v[60:63], v[80:83]\n\t"
        "v_mfma_f32_16x16x32_bf16 v[84:87], a[116:119], v[60:63], v[84:87]\n\t"
        "v_mfma_f32_16x16x32_bf16 v[88:91], a[120:123], v[60:63], v[88:91]\n\t"
        "v_mfma_f32_16x16x32_bf16 v[92:95], a[124:127], v[60:63], v[92:95]\n\t"
        "buffer_load_dwordx4 v5, s[8:11], s21 offen offset:2048 lds\n\t"
        "buffer_load_dwordx4 v24, s[8:11], s22 offen offset:3072 lds\n\t"
        "s_bfe_u32 s29, s46, 0x80018\n\t"
        "v_lshl_add_u32 v5, s29, 17, v2\n\t"
        "s_add_u32 m0, s37, 4096\n\t"
        "v_xor_b32 v24, 64, v5\n\t"
        "buffer_load_dwordx4 v5, s[8:11], 0 offen offset:0 lds\n\t"
        "buffer_load_dwordx4 v24, s[8:11], s20 offen offset:1024 lds\n\t"
        "s_waitcnt vmcnt(12)\n\t"
        "ds_read_b128 v[32:35], v136\n\t"
        "ds_read_b64_tr_b16 a[128:129], v8 offset:12288\n\t"
        "ds_read_b64_tr_b16 a[130:131], v8 offset:12800\n\t"
        "ds_read_b64_tr_b16 a[132:133], v9 offset:12288\n\t"
        "ds_read_b64_tr_b16 a[134:135], v9 offset:12800\n\t"
        "ds_read_b64_tr_b16 a[136:137], v10 offset:12288\n\t"
        "ds_read_b64_tr_b16 a[138:139], v10 offset:12800\n\t"
        "ds_read_b64_tr_b16 a[140:141], v11 offset:12288\n\t"
        "ds_read_b64_tr_b16 a[142:143], v11 offset:12800\n\t"
        "buffer_load_dwordx4 v5, s[8:11], s21 offen offset:2048 lds\n\t"
        "buffer_load_dwordx4 v24, s[8:11], s22 offen offset:3072 lds\n\t"
        "s_waitcnt lgkmcnt(0)\n\t"
        "v_mfma_f32_16x16x32_bf16 v[80:83], a[128:131], v[32:35], v[80:83]\n\t"
        "v_mfma_f32_16x16x32_bf16 v[84:87], a[132:135], v[32:35], v[84:87]\n\t"
        "v_mfma_f32_16x16x32_bf16 v[88:91], a[136:139], v[32:35], v[88:91]\n\t"
        "v_mfma_f32_16x16x32_bf16 v[92:95], a[140:143], v[32:35], v[92:95]\n\t"
        "s_barrier\n\t"
        "s_bfe_u32 s29, s30, 0x30018\n\t"
        "v_lshl_add_u32 v3, s29, 9, v1\n\t"
        "s_add_u32 m0, s38, 12288\n\t"
        "v_xor_b32 v4, 64, v3\n\t"
        "buffer_load_dwordx4 v3, s[4:7], 0 offen lds\n\t"
        "buffer_load_dwordx4 v4, s[4:7], s20 offen offset:1024 lds\n\t"
        "s_waitcnt vmcnt(12)\n\t"
        "ds_read_b128 v[36:39], v137\n\t"
        "ds_read_b64_tr_b16 a[144:145], v8 offset:16384\n\t"
        "ds_read_b64_tr_b16 a[146:147], v8 offset:16896\n\t"
        "ds_read_b64_tr_b16 a[148:149], v9 offset:16384\n\t"
        "ds_read_b64_tr_b16 a[150:151], v9 offset:16896\n\t"
        "ds_read_b64_tr_b16 a[152:153], v10 offset:16384\n\t"
        "ds_read_b64_tr_b16 a[154:155], v10 offset:16896\n\t"
        "ds_read_b64_tr_b16 a[156:157], v11 offset:16384\n\t"
        "ds_read_b64_tr_b16 a[158:159], v11 offset:16896\n\t"
        "s_bfe_u32 s29, s47, 0x80000\n\t"
        "v_lshl_add_u32 v5, s29, 17, v2\n\t"
        "s_add_u32 m0, s37, 8192\n\t"
        "v_xor_b32 v24, 64, v5\n\t"
        "buffer_load_dwordx4 v5, s[8:11], 0 offen offset:0 lds\n\t"
        "buffer_load_dwordx4 v24, s[8:11], s20 offen offset:1024 lds\n\t"
        "s_waitcnt lgkmcnt(0)\n\t"
        "v_mfma_f32_16x16x32_bf16 v[80:83], a[144:147], v[36:39], v[80:83]\n\t"
        "v_mfma_f32_16x16x32_bf16 v[84:87], a[148:151], v[36:39], v[84:87]\n\t"
        "v_mfma_f32_16x16x32_bf16 v[88:91], a[152:155], v[36:39], v[88:91]\n\t"
        "v_mfma_f32_16x16x32_bf16 v[92:95], a[156:159], v[36:39], v[92:95]\n\t"
        "buffer_load_dwordx4 v5, s[8:11], s21 offen offset:2048 lds\n\t"
        "buffer_load_dwordx4 v24, s[8:11], s22 offen offset:3072 lds\n\t"
        "s_bfe_u32 s29, s47, 0x80008\n\t"
        "v_lshl_add_u32 v5, s29, 17, v2\n\t"
        "s_add_u32 m0, s37, 12288\n\t"
        "v_xor_b32 v24, 64, v5\n\t"
        "buffer_load_dwordx4 v5, s[8:11], 0 offen offset:0 lds\n\t"
        "buffer_load_dwordx4 v24, s[8:11], s20 offen offset:1024 lds\n\t"
        "s_waitcnt vmcnt(12)\n\t"
        "ds_read_b128 v[40:43], v138\n\t"
        "ds_read_b64_tr_b16 a[160:161], v8 offset:0\n\t"
        "ds_read_b64_tr_b16 a[162:163], v8 offset:512\n\t"
        "ds_read_b64_tr_b16 a[164:165], v9 offset:0\n\t"
        "ds_read_b64_tr_b16 a[166:167], v9 offset:512\n\t"
        "ds_read_b64_tr_b16 a[168:169], v10 offset:0\n\t"
        "ds_read_b64_tr_b16 a[170:171], v10 offset:512\n\t"
        "ds_read_b64_tr_b16 a[172:173], v11 offset:0\n\t"
        "ds_read_b64_tr_b16 a[174:175], v11 offset:512\n\t"
        "buffer_load_dwordx4 v5, s[8:11], s21 offen offset:2048 lds\n\t"
        "buffer_load_dwordx4 v24, s[8:11], s22 offen offset:3072 lds\n\t"
        "s_waitcnt lgkmcnt(0)\n\t"
        "v_mfma_f32_16x16x32_bf16 v[80:83], a[160:163], v[40:43], v[80:83]\n\t"
        "v_mfma_f32_16x16x32_bf16 v[84:87], a[164:167], v[40:43], v[84:87]\n\t"
        "v_mfma_f32_16x16x32_bf16 v[88:91], a[168:171], v[40:43], v[88:91]\n\t"
        "v_mfma_f32_16x16x32_bf16 v[92:95], a[172:175], v[40:43], v[92:95]\n\t"
        "s_barrier\n\t"
        "s_bfe_u32 s29, s30, 0x3001c\n\t"
        "v_lshl_add_u32 v3, s29, 9, v1\n\t"
        "s_add_u32 m0, s38, 14336\n\t"
        "v_xor_b32 v4, 64, v3\n\t"
        "buffer_load_dwordx4 v3, s[4:7], 0 offen lds\n\t"
        "buffer_load_dwordx4 v4, s[4:7], s20 offen offset:1024 lds\n\t"
        "s_waitcnt vmcnt(12)\n\t"
        "ds_read_b128 v[44:47], v139\n\t"
        "ds_read_b64_tr_b16 a[176:177], v8 offset:4096\n\t"
        "ds_read_b64_tr_b16 a[178:179], v8 offset:4608\n\t"
        "ds_read_b64_tr_b16 a[180:181], v9 offset:4096\n\t"
        "ds_read_b64_tr_b16 a[182:183], v9 offset:4608\n\t"
        "ds_read_b64_tr_b16 a[184:185], v10 offset:4096\n\t"
        "ds_read_b64_tr_b16 a[186:187], v10 offset:4608\n\t"
        "ds_read_b64_tr_b16 a[188:189], v11 offset:4096\n\t"
        "ds_read_b64_tr_b16 a[190:191], v11 offset:4608\n\t"
        "s_bfe_u32 s29, s47, 0x80010\n\t"
        "v_lshl_add_u32 v5, s29, 17, v2\n\t"
        "s_add_u32 m0, s37, 16384\n\t"
        "v_xor_b32 v24, 64, v5\n\t"
        "buffer_load_dwordx4 v5, s[8:11], 0 offen offset:0 lds\n\t"
        "buffer_load_dwordx4 v24, s[8:11], s20 offen offset:1024 lds\n\t"
        "s_waitcnt lgkmcnt(0)\n\t"
        "v_mfma_f32_16x16x32_bf16 v[80:83], a[176:179], v[44:47], v[80:83]\n\t"
        "v_mfma_f32_16x16x32_bf16 v[84:87], a[180:183], v[44:47], v[84:87]\n\t"
        "v_mfma_f32_16x16x32_bf16 v[88:91], a[184:187], v[44:47], v[88:91]\n\t"
        "v_mfma_f32_16x16x32_bf16 v[92:95], a[188:191], v[44:47], v[92:95]\n\t"
        "buffer_load_dwordx4 v5, s[8:11], s21 offen offset:2048 lds\n\t"
        "buffer_load_dwordx4 v24, s[8:11], s22 offen offset:3072 lds\n\t"
        "s_bfe_u32 s29, s47, 0x80018\n\t"
        "v_lshl_add_u32 v5, s29, 17, v2\n\t"
        "s_mov_b32 m0, s37\n\t"
        "v_xor_b32 v24, 64, v5\n\t"
        "buffer_load_dwordx4 v5, s[8:11], 0 offen offset:0 lds\n\t"
        "buffer_load_dwordx4 v24, s[8:11], s20 offen offset:1024 lds\n\t"
        "s_waitcnt vmcnt(12)\n\t"
        "ds_read_b128 v[48:51], v140\n\t"
        "ds_read_b64_tr_b16 a[192:193], v8 offset:8192\n\t"
        "ds_read_b64_tr_b16 a[194:195], v8 offset:8704\n\t"
        "ds_read_b64_tr_b16 a[196:197], v9 offset:8192\n\t"
        "ds_read_b64_tr_b16 a[198:199], v9 offset:8704\n\t"
        "ds_read_b64_tr_b16 a[200:201], v10 offset:8192\n\t"
        "ds_read_b64_tr_b16 a[202:203], v10 offset:8704\n\t"
        "ds_read_b64_tr_b16 a[204:205], v11 offset:8192\n\t"
        "ds_read_b64_tr_b16 a[206:207], v11 offset:8704\n\t"
        "buffer_load_dwordx4 v5, s[8:11], s21 offen offset:2048 lds\n\t"
        "buffer_load_dwordx4 v24, s[8:11], s22 offen offset:3072 lds\n\t"
        "s_waitcnt lgkmcnt(0)\n\t"
        "v_mfma_f32_16x16x32_bf16 v[80:83], a[192:195], v[48:51], v[80:83]\n\t"
        "v_mfma_f32_16x16x32_bf16 v[84:87], a[196:199], v[48:51], v[84:87]\n\t"
        "v_mfma_f32_16x16x32_bf16 v[88:91], a[200:203], v[48:51], v[88:91]\n\t"
        "v_mfma_f32_16x16x32_bf16 v[92:95], a[204:207], v[48:51], v[92:95]\n\t"
        "s_barrier\n\t"
        "s_bfe_u32 s29, s30, 0x30000\n\t"
        "v_lshl_add_u32 v3, s29, 9, v25\n\t"
        "s_mov_b32 m0, s38\n\t"
        "v_xor_b32 v4, 64, v3\n\t"
        "buffer_load_dwordx4 v3, s[4:7], 0 offen lds\n\t"
        "buffer_load_dwordx4 v4, s[4:7], s20 offen offset:1024 lds\n\t"
        "s_waitcnt vmcnt(12)\n\t"
        "ds_read_b128 v[52:55], v141\n\t"
        "ds_read_b64_tr_b16 a[208:209], v8 offset:12288\n\t"
        "ds_read_b64_tr_b16 a[210:211], v8 offset:12800\n\t"
        "ds_read_b64_tr_b16 a[212:213], v9 offset:12288\n\t"
        "ds_read_b64_tr_b16 a[214:215], v9 offset:12800\n\t"
        "ds_read_b64_tr_b16 a[216:217], v10 offset:12288\n\t"
        "ds_read_b64_tr_b16 a[218:219], v10 offset:12800\n\t"
        "ds_read_b64_tr_b16 a[220:221], v11 offset:12288\n\t"
        "ds_read_b64_tr_b16 a[222:223], v11 offset:12800\n\t"
        "s_bfe_u32 s29, s30, 0x30004\n\t"
        "v_lshl_add_u32 v3, s29, 9, v25\n\t"
        "s_add_u32 m0, s38, 2048\n\t"
        "v_xor_b32 v4, 64, v3\n\t"
        "buffer_load_dwordx4 v3, s[4:7], 0 offen lds\n\t"
        "buffer_load_dwordx4 v4, s[4:7], s20 offen offset:1024 lds\n\t"
        "s_waitcnt lgkmcnt(0)\n\t"
        "v_mfma_f32_16x16x32_bf16 v[80:83], a[208:211], v[52:55], v[80:83]\n\t"
        "v_mfma_f32_16x16x32_bf16 v[84:87], a[212:215], v[52:55], v[84:87]\n\t"
        "v_mfma_f32_16x16x32_bf16 v[88:91], a[216:219], v[52:55], v[88:91]\n\t"
        "v_mfma_f32_16x16x32_bf16 v[92:95], a[220:223], v[52:55], v[92:95]\n\t"
        "s_bfe_u32 s29, s30, 0x30008\n\t"
        "v_lshl_add_u32 v3, s29, 9, v25\n\t"
        "s_add_u32 m0, s38, 4096\n\t"
        "v_xor_b32 v4, 64, v3\n\t"
        "buffer_load_dwordx4 v3, s[4:7], 0 offen lds\n\t"
        "buffer_load_dwordx4 v4, s[4:7], s20 offen offset:1024 lds\n\t"
        "s_waitcnt vmcnt(10)\n\t"
        "ds_read_b128 v[56:59], v142\n\t"
        "ds_read_b128 v[60:63], v143\n\t"
        "ds_read_b64_tr_b16 a[224:225], v8 offset:16384\n\t"
        "ds_read_b64_tr_b16 a[226:227], v8 offset:16896\n\t"
        "ds_read_b64_tr_b16 a[228:229], v9 offset:16384\n\t"
        "ds_read_b64_tr_b16 a[230:231], v9 offset:16896\n\t"
        "ds_read_b64_tr_b16 a[232:233], v10 offset:16384\n\t"
        "ds_read_b64_tr_b16 a[234:235], v10 offset:16896\n\t"
        "ds_read_b64_tr_b16 a[236:237], v11 offset:16384\n\t"
        "ds_read_b64_tr_b16 a[238:239], v11 offset:16896\n\t"
        "s_waitcnt lgkmcnt(0)\n\t"
        "v_mfma_f32_16x16x32_bf16 v[80:83], a[224:227], v[56:59], v[80:83]\n\t"
        "v_mfma_f32_16x16x32_bf16 v[84:87], a[228:231], v[56:59], v[84:87]\n\t"
        "v_mfma_f32_16x16x32_bf16 v[88:91], a[232:235], v[56:59], v[88:91]\n\t"
        "v_mfma_f32_16x16x32_bf16 v[92:95], a[236:239], v[56:59], v[92:95]\n\t"
        "s_waitcnt vmcnt(6)\n\t"
        "ds_read_b64_tr_b16 a[240:241], v8 offset:0\n\t"
        "ds_read_b64_tr_b16 a[242:243], v8 offset:512\n\t"
        "ds_read_b64_tr_b16 a[244:245], v9 offset:0\n\t"
        "ds_read_b64_tr_b16 a[246:247], v9 offset:512\n\t"
        "ds_read_b64_tr_b16 a[248:249], v10 offset:0\n\t"
        "ds_read_b64_tr_b16 a[250:251], v10 offset:512\n\t"
        "ds_read_b64_tr_b16 a[252:253], v11 offset:0\n\t"
        "ds_read_b64_tr_b16 a[254:255], v11 offset:512\n\t"
        "s_waitcnt lgkmcnt(6)\n\t"
        "v_mfma_f32_16x16x32_bf16 v[80:83], a[240:243], v[60:63], v[80:83]\n\t"
        "s_waitcnt lgkmcnt(4)\n\t"
        "v_mfma_f32_16x16x32_bf16 v[84:87], a[244:247], v[60:63], v[84:87]\n\t"
        "s_waitcnt lgkmcnt(2)\n\t"
        "v_mfma_f32_16x16x32_bf16 v[88:91], a[248:251], v[60:63], v[88:91]\n\t"
        "s_waitcnt lgkmcnt(0)\n\t"
        "v_mfma_f32_16x16x32_bf16 v[92:95], a[252:255], v[60:63], v[92:95]\n\t"
        "s_nop 4\n\t"
        "ds_write_b128 v12, v[80:83] offset:0\n\t"
        "ds_write_b128 v13, v[84:87] offset:0\n\t"
        "ds_write_b128 v12, v[88:91] offset:128\n\t"
        "ds_write_b128 v13, v[92:95] offset:128\n\t"
        "s_bfe_u32 s29, s30, 0x3000c\n\t"
        "v_lshl_add_u32 v3, s29, 9, v25\n\t"
        "s_add_u32 m0, s38, 6144\n\t"
        "v_xor_b32 v4, 64, v3\n\t"
        "buffer_load_dwordx4 v3, s[4:7], 0 offen lds\n\t"
        "buffer_load_dwordx4 v4, s[4:7], s20 offen offset:1024 lds\n\t"
        "s_bfe_u32 s29, s30, 0x30010\n\t"
        "v_lshl_add_u32 v3, s29, 9, v25\n\t"
        "s_add_u32 m0, s38, 8192\n\t"
        "v_xor_b32 v4, 64, v3\n\t"
        "buffer_load_dwordx4 v3, s[4:7], 0 offen lds\n\t"
        "buffer_load_dwordx4 v4, s[4:7], s20 offen offset:1024 lds\n\t"
        "s_waitcnt lgkmcnt(0)\n\t"
        "s_barrier\n\t"
        "ds_read_b128 v[64:67], v16 offset:0\n\t"
        "ds_read_b128 v[68:71], v16 offset:4096\n\t"
        "ds_read_b128 v[72:75], v16 offset:8192\n\t"
        "ds_read_b128 v[76:79], v16 offset:12288\n\t"
        "s_bfe_u32 s29, s30, 0x30014\n\t"
        "v_lshl_add_u32 v3, s29, 9, v25\n\t"
        "s_add_u32 m0, s38, 10240\n\t"
        "v_xor_b32 v4, 64, v3\n\t"
        "buffer_load_dwordx4 v3, s[4:7], 0 offen lds\n\t"
        "buffer_load_dwordx4 v4, s[4:7], s20 offen offset:1024 lds\n\t"
        "s_bfe_u32 s29, s30, 0x30018\n\t"
        "v_lshl_add_u32 v3, s29, 9, v25\n\t"
        "s_add_u32 m0, s38, 12288\n\t"
        "v_xor_b32 v4, 64, v3\n\t"
        "buffer_load_dwordx4 v3, s[4:7], 0 offen lds\n\t"
        "buffer_load_dwordx4 v4, s[4:7], s20 offen offset:1024 lds\n\t"
        "s_waitcnt lgkmcnt(2)\n\t"
        "v_pk_add_f32 v[64:65], v[64:65], v[68:69]\n\t"
        "v_pk_add_f32 v[66:67], v[66:67], v[70:71]\n\t"
        "s_waitcnt lgkmcnt(0)\n\t"
        "v_pk_add_f32 v[72:73], v[72:73], v[76:77]\n\t"
        "v_pk_add_f32 v[74:75], v[74:75], v[78:79]\n\t"
        "v_pk_add_f32 v[64:65], v[64:65], v[72:73]\n\t"
        "v_pk_add_f32 v[66:67], v[66:67], v[74:75]\n\t"
        "s_bfe_u32 s29, s30, 0x3001c\n\t"
        "v_lshl_add_u32 v3, s29, 9, v25\n\t"
        "s_add_u32 m0, s38, 14336\n\t"
        "v_xor_b32 v4, 64, v3\n\t"
        "buffer_load_dwordx4 v3, s[4:7], 0 offen lds\n\t"
        "buffer_load_dwordx4 v4, s[4:7], s20 offen offset:1024 lds\n\t"
        "v_pk_mul_f32 v[26:27], v[64:65], s[40:41]\n\t"
        "v_pk_mul_f32 v[28:29], v[66:67], s[40:41]\n\t"
        "v_med3_f32 v26, v26, -s42, s42\n\t"
        "v_med3_f32 v27, v27, -s42, s42\n\t"
        "v_med3_f32 v28, v28, -s42, s42\n\t"
        "v_med3_f32 v29, v29, -s42, s42\n\t"
        "v_exp_f32 v26, v26\n\t"
        "v_exp_f32 v27, v27\n\t"
        "v_exp_f32 v28, v28\n\t"
        "v_exp_f32 v29, v29\n\t"
        "s_nop 0\n\t"
        "v_pk_add_f32 v[26:27], v[26:27], 1.0 op_sel_hi:[1,0]\n\t"
        "v_pk_add_f32 v[28:29], v[28:29], 1.0 op_sel_hi:[1,0]\n\t"
        "v_rcp_f32 v26, v26\n\t"
        "v_rcp_f32 v27, v27\n\t"
        "v_rcp_f32 v28, v28\n\t"
        "v_rcp_f32 v29, v29\n\t"
        "s_nop 0\n\t"
        "v_pk_mul_f32 v[26:27], v[26:27], v[64:65]\n\t"
        "v_pk_mul_f32 v[28:29], v[28:29], v[66:67]\n\t"
        "v_cvt_pk_bf16_f32 v112, v26, v27\n\t"
        "v_cvt_pk_bf16_f32 v113, v28, v29\n\t"
        "s_nop 1\n\t"
        "v_permlane32_swap_b32 v112, v114\n\t"
        "v_permlane32_swap_b32 v113, v115\n\t"
        "s_mov_b32 exec_hi, 0\n\t"
        "global_store_dwordx4 v18, v[112:115], s[52:53] sc0 sc1\n\t"
        "s_mov_b32 exec_hi, -1\n\t"
        "s_waitcnt vmcnt(15)\n\t"
        "ds_read_b128 v[32:35], v128\n\t"
        "s_waitcnt vmcnt(13)\n\t"
        "ds_read_b128 v[36:39], v129\n\t"
        "s_waitcnt vmcnt(11)\n\t"
        "ds_read_b128 v[40:43], v130\n\t"
        "s_waitcnt lgkmcnt(2)\n\t"
        "v_mfma_f32_16x16x32_bf16 v[96:99], a[0:3], v[32:35], 0\n\t"
        "v_mfma_f32_16x16x32_bf16 v[100:103], a[4:7], v[32:35], 0\n\t"
        "v_mfma_f32_16x16x32_bf16 v[104:107], a[8:11], v[32:35], 0\n\t"
        "v_mfma_f32_16x16x32_bf16 v[108:111], a[12:15], v[32:35], 0\n\t"
        "s_waitcnt vmcnt(9)\n\t"
        "ds_read_b128 v[44:47], v131\n\t"
        "s_waitcnt lgkmcnt(2)\n\t"
        "v_mfma_f32_16x16x32_bf16 v[96:99], a[16:19], v[36:39], v[96:99]\n\t"
        "v_mfma_f32_16x16x32_bf16 v[100:103], a[20:23], v[36:39], v[100:103]\n\t"
        "v_mfma_f32_16x16x32_bf16 v[104:107], a[24:27], v[36:39], v[104:107]\n\t"
        "v_mfma_f32_16x16x32_bf16 v[108:111], a[28:31], v[36:39], v[108:111]\n\t"
        "ds_read_b128 v[48:51], v132\n\t"
        "s_waitcnt lgkmcnt(2)\n\t"
        "v_mfma_f32_16x16x32_bf16 v[96:99], a[32:35], v[40:43], v[96:99]\n\t"
        "v_mfma_f32_16x16x32_bf16 v[100:103], a[36:39], v[40:43], v[100:103]\n\t"
        "v_mfma_f32_16x16x32_bf16 v[104:107], a[40:43], v[40:43], v[104:107]\n\t"
        "v_mfma_f32_16x16x32_bf16 v[108:111], a[44:47], v[40:43], v[108:111]\n\t"
        "ds_read_b128 v[52:55], v133\n\t"
        "s_waitcnt lgkmcnt(2)\n\t"
        "v_mfma_f32_16x16x32_bf16 v[96:99], a[48:51], v[44:47], v[96:99]\n\t"
        "v_mfma_f32_16x16x32_bf16 v[100:103], a[52:55], v[44:47], v[100:103]\n\t"
        "v_mfma_f32_16x16x32_bf16 v[104:107], a[56:59], v[44:47], v[104:107]\n\t"
        "v_mfma_f32_16x16x32_bf16 v[108:111], a[60:63], v[44:47], v[108:111]\n\t"
        "ds_read_b128 v[56:59], v134\n\t"
        "s_waitcnt lgkmcnt(2)\n\t"
        "v_mfma_f32_16x16x32_bf16 v[96:99], a[64:67], v[48:51], v[96:99]\n\t"
        "v_mfma_f32_16x16x32_bf16 v[100:103], a[68:71], v[48:51], v[100:103]\n\t"
        "v_mfma_f32_16x16x32_bf16 v[104:107], a[72:75], v[48:51], v[104:107]\n\t"
        "v_mfma_f32_16x16x32_bf16 v[108:111], a[76:79], v[48:51], v[108:111]\n\t"
        "ds_read_b128 v[60:63], v135\n\t"
        "s_waitcnt lgkmcnt(2)\n\t"
        "v_mfma_f32_16x16x32_bf16 v[96:99], a[80:83], v[52:55], v[96:99]\n\t"
        "v_mfma_f32_16x16x32_bf16 v[100:103], a[84:87], v[52:55], v[100:103]\n\t"
        "v_mfma_f32_16x16x32_bf16 v[104:107], a[88:91], v[52:55], v[104:107]\n\t"
        "v_mfma_f32_16x16x32_bf16 v[108:111], a[92:95], v[52:55], v[108:111]\n\t"
        "s_waitcnt vmcnt(7)\n\t"
        "ds_read_b128 v[32:35], v136\n\t"
        "s_waitcnt lgkmcnt(2)\n\t"
        "v_mfma_f32_16x16x32_bf16 v[96:99], a[96:99], v[56:59], v[96:99]\n\t"
        "v_mfma_f32_16x16x32_bf16 v[100:103], a[100:103], v[56:59], v[100:103]\n\t"
        "v_mfma_f32_16x16x32_bf16 v[104:107], a[104:107], v[56:59], v[104:107]\n\t"
        "v_mfma_f32_16x16x32_bf16 v[108:111], a[108:111], v[56:59], v[108:111]\n\t"
        "ds_read_b128 v[36:39], v137\n\t"
        "s_waitcnt lgkmcnt(2)\n\t"
        "v_mfma_f32_16x16x32_bf16 v[96:99], a[112:115], v[60:63], v[96:99]\n\t"
        "v_mfma_f32_16x16x32_bf16 v[100:103], a[116:119], v[60:63], v[100:103]\n\t"
        "v_mfma_f32_16x16x32_bf16 v[104:107], a[120:123], v[60:63], v[104:107]\n\t"
        "v_mfma_f32_16x16x32_bf16 v[108:111], a[124:127], v[60:63], v[108:111]\n\t"
        "s_waitcnt vmcnt(5)\n\t"
        "ds_read_b128 v[40:43], v138\n\t"
        "s_waitcnt lgkmcnt(2)\n\t"
        "v_mfma_f32_16x16x32_bf16 v[96:99], a[128:131], v[32:35], v[96:99]\n\t"
        "v_mfma_f32_16x16x32_bf16 v[100:103], a[132:135], v[32:35], v[100:103]\n\t"
        "v_mfma_f32_16x16x32_bf16 v[104:107], a[136:139], v[32:35], v[104:107]\n\t"
        "v_mfma_f32_16x16x32_bf16 v[108:111], a[140:143], v[32:35], v[108:111]\n\t"
        "ds_read_b128 v[44:47], v139\n\t"
        "s_waitcnt lgkmcnt(2)\n\t"
        "v_mfma_f32_16x16x32_bf16 v[96:99], a[144:147], v[36:39], v[96:99]\n\t"
        "v_mfma_f32_16x16x32_bf16 v[100:103], a[148:151], v[36:39], v[100:103]\n\t"
        "v_mfma_f32_16x16x32_bf16 v[104:107], a[152:155], v[36:39], v[104:107]\n\t"
        "v_mfma_f32_16x16x32_bf16 v[108:111], a[156:159], v[36:39], v[108:111]\n\t"
        "s_waitcnt vmcnt(3)\n\t"
        "ds_read_b128 v[48:51], v140\n\t"
        "s_waitcnt lgkmcnt(2)\n\t"
        "v_mfma_f32_16x16x32_bf16 v[96:99], a[160:163], v[40:43], v[96:99]\n\t"
        "v_mfma_f32_16x16x32_bf16 v[100:103], a[164:167], v[40:43], v[100:103]\n\t"
        "v_mfma_f32_16x16x32_bf16 v[104:107], a[168:171], v[40:43], v[104:107]\n\t"
        "v_mfma_f32_16x16x32_bf16 v[108:111], a[172:175], v[40:43], v[108:111]\n\t"
        "ds_read_b128 v[52:55], v141\n\t"
        "s_waitcnt lgkmcnt(2)\n\t"
        "v_mfma_f32_16x16x32_bf16 v[96:99], a[176:179], v[44:47], v[96:99]\n\t"
        "v_mfma_f32_16x16x32_bf16 v[100:103], a[180:183], v[44:47], v[100:103]\n\t"
        "v_mfma_f32_16x16x32_bf16 v[104:107], a[184:187], v[44:47], v[104:107]\n\t"
        "v_mfma_f32_16x16x32_bf16 v[108:111], a[188:191], v[44:47], v[108:111]\n\t"
        "s_waitcnt vmcnt(1)\n\t"
        "ds_read_b128 v[56:59], v142\n\t"
        "s_waitcnt lgkmcnt(2)\n\t"
        "v_mfma_f32_16x16x32_bf16 v[96:99], a[192:195], v[48:51], v[96:99]\n\t"
        "v_mfma_f32_16x16x32_bf16 v[100:103], a[196:199], v[48:51], v[100:103]\n\t"
        "v_mfma_f32_16x16x32_bf16 v[104:107], a[200:203], v[48:51], v[104:107]\n\t"
        "v_mfma_f32_16x16x32_bf16 v[108:111], a[204:207], v[48:51], v[108:111]\n\t"
        "ds_read_b128 v[60:63], v143\n\t"
        "s_waitcnt lgkmcnt(2)\n\t"
        "v_mfma_f32_16x16x32_bf16 v[96:99], a[208:211], v[52:55], v[96:99]\n\t"
        "v_mfma_f32_16x16x32_bf16 v[100:103], a[212:215], v[52:55], v[100:103]\n\t"
        "v_mfma_f32_16x16x32_bf16 v[104:107], a[216:219], v[52:55], v[104:107]\n\t"
        "v_mfma_f32_16x16x32_bf16 v[108:111], a[220:223], v[52:55], v[108:111]\n\t"
        "s_waitcnt lgkmcnt(1)\n\t"
        "v_mfma_f32_16x16x32_bf16 v[96:99], a[224:227], v[56:59], v[96:99]\n\t"
        "v_mfma_f32_16x16x32_bf16 v[100:103], a[228:231], v[56:59], v[100:103]\n\t"
        "v_mfma_f32_16x16x32_bf16 v[104:107], a[232:235], v[56:59], v[104:107]\n\t"
        "v_mfma_f32_16x16x32_bf16 v[108:111], a[236:239], v[56:59], v[108:111]\n\t"
        "s_waitcnt lgkmcnt(0)\n\t"
        "v_mfma_f32_16x16x32_bf16 v[96:99], a[240:243], v[60:63], v[96:99]\n\t"
        "v_mfma_f32_16x16x32_bf16 v[100:103], a[244:247], v[60:63], v[100:103]\n\t"
        "v_mfma_f32_16x16x32_bf16 v[104:107], a[248:251], v[60:63], v[104:107]\n\t"
        "v_mfma_f32_16x16x32_bf16 v[108:111], a[252:255], v[60:63], v[108:111]\n\t"
        "s_nop 4\n\t"
        "ds_write_b128 v12, v[96:99] offset:16384\n\t"
        "ds_write_b128 v13, v[100:103] offset:16384\n\t"
        "ds_write_b128 v12, v[104:107] offset:16512\n\t"
        "ds_write_b128 v13, v[108:111] offset:16512\n\t"
        "s_waitcnt lgkmcnt(0)\n\t"
        "s_barrier\n\t"
        "ds_read_b128 v[64:67], v16 offset:16384\n\t"
        "ds_read_b128 v[68:71], v16 offset:20480\n\t"
        "ds_read_b128 v[72:75], v16 offset:24576\n\t"
        "ds_read_b128 v[76:79], v16 offset:28672\n\t"
        "s_waitcnt lgkmcnt(2)\n\t"
        "v_pk_add_f32 v[64:65], v[64:65], v[68:69]\n\t"
        "v_pk_add_f32 v[66:67], v[66:67], v[70:71]\n\t"
        "s_waitcnt lgkmcnt(0)\n\t"
        "v_pk_add_f32 v[72:73], v[72:73], v[76:77]\n\t"
        "v_pk_add_f32 v[74:75], v[74:75], v[78:79]\n\t"
        "v_pk_add_f32 v[64:65], v[64:65], v[72:73]\n\t"
        "v_pk_add_f32 v[66:67], v[66:67], v[74:75]\n\t"
        "v_pk_mul_f32 v[26:27], v[64:65], s[40:41]\n\t"
        "v_pk_mul_f32 v[28:29], v[66:67], s[40:41]\n\t"
        "v_med3_f32 v26, v26, -s42, s42\n\t"
        "v_med3_f32 v27, v27, -s42, s42\n\t"
        "v_med3_f32 v28, v28, -s42, s42\n\t"
        "v_med3_f32 v29, v29, -s42, s42\n\t"
        "v_exp_f32 v26, v26\n\t"
        "v_exp_f32 v27, v27\n\t"
        "v_exp_f32 v28, v28\n\t"
        "v_exp_f32 v29, v29\n\t"
        "s_nop 0\n\t"
        "v_pk_add_f32 v[26:27], v[26:27], 1.0 op_sel_hi:[1,0]\n\t"
        "v_pk_add_f32 v[28:29], v[28:29], 1.0 op_sel_hi:[1,0]\n\t"
        "v_rcp_f32 v26, v26\n\t"
        "v_rcp_f32 v27, v27\n\t"
        "v_rcp_f32 v28, v28\n\t"
        "v_rcp_f32 v29, v29\n\t"
        "s_nop 0\n\t"
        "v_pk_mul_f32 v[26:27], v[26:27], v[64:65]\n\t"
        "v_pk_mul_f32 v[28:29], v[28:29], v[66:67]\n\t"
        "v_cvt_pk_bf16_f32 v116, v26, v27\n\t"
        "v_cvt_pk_bf16_f32 v117, v28, v29\n\t"
        "s_nop 1\n\t"
        "v_permlane32_swap_b32 v116, v118\n\t"
        "v_permlane32_swap_b32 v117, v119\n\t"
        "s_mov_b32 exec_hi, 0\n\t"
        "global_store_dwordx4 v19, v[116:119], s[52:53] sc0 sc1\n\t"
        "s_mov_b32 exec_hi, -1\n\t"
        "s_cmp_lg_u32 s27, 0\n\t"
        "s_mov_b64 exec, 1\n\t"
        "s_waitcnt vmcnt(0)\n\t"
        "s_barrier\n\t"
        "s_cbranch_scc1 9f\n\t"
        "global_store_dword v21, v20, s[54:55] sc0 sc1\n\t"
        "global_store_dword v22, v20, s[12:13] sc0 sc1\n\t"
        "s_mov_b64 exec, -1\n\t"
        "s_cmp_lg_u32 s35, s14\n\t"
        "s_cbranch_scc1 9f\n\t"
        "v_lshl_add_u32 v23, %[tid], 7, v21\n\t"
        "v_add_u32 v23, 16384, v23\n\t"
        "v_writelane_b32 v23, s25, 8\n\t"
        "s_mov_b64 exec, 0x1ff\n\t"
        "s_memrealtime s[44:45]\n\t"
        "s_waitcnt lgkmcnt(0)\n\t"
        "s_add_u32 s44, s44, 50000000\n\t"
        "s_addc_u32 s45, s45, 0\n\t"
        "2:\n\t"
        "global_load_dword v32, v23, s[12:13] sc0 sc1\n\t"
        "s_memrealtime s[46:47]\n\t"
        "s_waitcnt vmcnt(0)\n\t"
        "v_cmp_gt_u32 vcc, s31, v32\n\t"
        "s_and_b32 s29, vcc_lo, 0xff\n\t"
        "s_cbranch_scc0 9f\n\t"
        "v_readlane_b32 s29, v32, 8\n\t"
        "s_cmp_eq_u32 s29, s31\n\t"
        "s_cbranch_scc1 8f\n\t"
        "s_waitcnt lgkmcnt(0)\n\t"
        "s_sub_u32 s46, s44, s46\n\t"
        "s_subb_u32 s47, s45, s47\n\t"
        "s_cbranch_scc0 2b\n\t"
        "8:\n\t"
        "s_mov_b64 exec, 1\n\t"
        "v_mov_b32 v23, 8192\n\t"
        "global_store_dword v23, v20, s[12:13] sc1\n\t"
        "v_mov_b32 v23, 0x7fc0\n\t"
        "global_store_short v18, v23, s[52:53] sc0 sc1\n\t"
        "9:\n\t"
        "s_mov_b64 exec, -1\n\t"
        "s_waitcnt lgkmcnt(0)\n\t"
        :
        : [ka] "s"(__builtin_amdgcn_kernarg_segment_ptr()), [bid] "s"(blockIdx.x), [tid] "v"(threadIdx.x),
          [lds] "i"((__attribute__((address_space(3))) char*)smem)
        : "v1", "v2", "v3", "v4", "v5", "v6", "v7", "v8", "v9", "v10", "v11", "v12", "v13", "v14", "v15", "v16", "v17",
          "v18", "v19", "v20", "v21", "v22", "v23", "v24", "v25", "v26", "v27", "v28", "v29", "v30", "v31", "v32", "v33",
          "v34", "v35", "v36", "v37", "v38", "v39", "v40", "v41", "v42", "v43", "v44", "v45", "v46", "v47", "v48", "v49",
          "v50", "v51", "v52", "v53", "v54", "v55", "v56", "v57", "v58", "v59", "v60", "v61", "v62", "v63", "v64", "v65",
          "v66", "v67", "v68", "v69", "v70", "v71", "v72", "v73", "v74", "v75", "v76", "v77", "v78", "v79", "v80", "v81",
          "v82", "v83", "v84", "v85", "v86", "v87", "v88", "v89", "v90", "v91", "v92", "v93", "v94", "v95", "v96", "v97",
          "v98", "v99", "v100", "v101", "v102", "v103", "v104", "v105", "v106", "v107", "v108", "v109", "v110", "v111",
          "v112", "v113", "v114", "v115", "v116", "v117", "v118", "v119", "v120", "v121", "v122", "v123", "v124", "v125",
          "v126", "v127", "v128", "v129", "v130", "v131", "v132", "v133", "v134", "v135", "v136", "v137", "v138", "v139",
          "v140", "v141", "v142", "v143", "a0", "a1", "a2", "a3", "a4", "a5", "a6", "a7", "a8", "a9", "a10", "a11",
          "a12", "a13", "a14", "a15", "a16", "a17", "a18", "a19", "a20", "a21", "a22", "a23", "a24", "a25", "a26", "a27",
          "a28", "a29", "a30", "a31", "a32", "a33", "a34", "a35", "a36", "a37", "a38", "a39", "a40", "a41", "a42", "a43",
          "a44", "a45", "a46", "a47", "a48", "a49", "a50", "a51", "a52", "a53", "a54", "a55", "a56", "a57", "a58", "a59",
          "a60", "a61", "a62", "a63", "a64", "a65", "a66", "a67", "a68", "a69", "a70", "a71", "a72", "a73", "a74", "a75",
          "a76", "a77", "a78", "a79", "a80", "a81", "a82", "a83", "a84", "a85", "a86", "a87", "a88", "a89", "a90", "a91",
          "a92", "a93", "a94", "a95", "a96", "a97", "a98", "a99", "a100", "a101", "a102", "a103", "a104", "a105", "a106",
          "a107", "a108", "a109", "a110", "a111", "a112", "a113", "a114", "a115", "a116", "a117", "a118", "a119", "a120",
          "a121", "a122", "a123", "a124", "a125", "a126", "a127", "a128", "a129", "a130", "a131", "a132", "a133", "a134",
          "a135", "a136", "a137", "a138", "a139", "a140", "a141", "a142", "a143", "a144", "a145", "a146", "a147", "a148",
          "a149", "a150", "a151", "a152", "a153", "a154", "a155", "a156", "a157", "a158", "a159", "a160", "a161", "a162",
          "a163", "a164", "a165", "a166", "a167", "a168", "a169", "a170", "a171", "a172", "a173", "a174", "a175", "a176",
          "a177", "a178", "a179", "a180", "a181", "a182", "a183", "a184", "a185", "a186", "a187", "a188", "a189", "a190",
          "a191", "a192", "a193", "a194", "a195", "a196", "a197", "a198", "a199", "a200", "a201", "a202", "a203", "a204",
          "a205", "a206", "a207", "a208", "a209", "a210", "a211", "a212", "a213", "a214", "a215", "a216", "a217", "a218",
          "a219", "a220", "a221", "a222", "a223", "a224", "a225", "a226", "a227", "a228", "a229", "a230", "a231", "a232",
          "a233", "a234", "a235", "a236", "a237", "a238", "a239", "a240", "a241", "a242", "a243", "a244", "a245", "a246",
          "a247", "a248", "a249", "a250", "a251", "a252", "a253", "a254", "a255", "s4", "s5", "s6", "s7", "s8", "s9",
          "s10", "s11", "s12", "s13", "s14", "s15", "s16", "s17", "s18", "s19", "s20", "s21", "s22", "s23", "s24", "s25",
          "s26", "s27", "s28", "s29", "s30", "s31", "s33", "s34", "s35", "s36", "s37", "s38", "s39", "s40", "s41", "s42",
          "s43", "s44", "s45", "s46", "s47", "s48", "s49", "s50", "s51", "s52", "s53", "s54", "s55", "s56", "s57", "s58",
          "s59", "s60", "s61", "s62", "s63", "s64", "s65", "s66", "s67", "s68", "s69", "s70", "s71", "s72", "s73", "s74",
          "s75", "vcc", "scc", "memory");
}

extern "C" void kernel_launch(int rank, int nranks, ncclComm_t comm,
                              hipStream_t stream,
                              void* const* d_in, const int* in_sizes, int n_in,
                              void* d_out, int out_size,
                              const int* in_dtypes, int out_dtype,
                              void* const* d_out_all,
                              void* const* d_ws_all, size_t ws_bytes) {
    if (nranks != NR || (unsigned)rank >= (unsigned)NR || n_in < 2 || in_sizes[0] != MS * KK ||
        in_sizes[1] != KK * NN || out_size != NN * NS || ws_bytes < WS_USED) {
        fprintf(stderr, "unexpected problem shape\n");
        return;
    }
    Args a;
    const uint64_t px = (uint64_t)d_in[0], pw = (uint64_t)d_in[1];
    a.rx[0] = (unsigned)px; a.rx[1] = (unsigned)(px >> 32) & 0xffffu; a.rx[2] = MS * KK * 2; a.rx[3] = 0x00020000u;
    a.rw[0] = (unsigned)pw; a.rw[1] = (unsigned)(pw >> 32) & 0xffffu; a.rw[2] = KK * NN * 2; a.rw[3] = 0x00020000u;
    const int cx = rank ^ LOCAL_GROUP;
    for (int g = 0; g < NR; ++g) {
        const unsigned p = (unsigned)(g ^ cx);
        if (p >= (unsigned)nranks) return;
        a.dst[g].out = (char*)d_out_all[p] + (size_t)rank * MS * NS * 2;
        a.dst[g].flag = (char*)d_ws_all[p] + WS_FLAG + rank * 128;
    }
    a.ws = (char*)d_ws_all[rank];
    a.waiter = LOCAL_GROUP;
    a.cx = cx << 9;
    a.slow = (int)((3 - (pw >> 7)) & 7) ^ ((cx & 1) << 2);
    a.pad[0] = a.pad[1] = a.pad[2] = 0;
    hipLaunchKernelGGL(gemm_a2a, dim3(256), dim3(256), 0, stream, a);
    HIP_CHECK(hipGetLastError());
}
```
